```python
import numpy as np

kernel_name = "gpu_a2a_gemm_m4096_k4096_n16384_bf16_none"


ABSMAX_RTOL = 0.008

COMPUTE_DTYPE = "bfloat16"
EPILOGUE = "none"
IN_QUANT = None
OUT_QUANT = None
IN_DTYPES = {"x": "bfloat16", "w_mat": "bfloat16"}
WORLD_SIZE = 8

def _to_bf16(x):
    u = np.ascontiguousarray(x, dtype=np.float32).view(np.uint32)
    return ((u + 0x7FFF + ((u >> 16) & 1)) & 0xFFFF0000).view(np.float32)

def setup_inputs():

    x = _to_bf16(np.random.randn(4096, 4096).astype(np.float32))
    w_mat = _to_bf16((np.random.randn(4096, 16384) / np.sqrt(4096)).astype(np.float32))
    return {"x": x, "w_mat": w_mat}

def _epilogue(y):
    return y

def reference(x, w_mat):
    y = (x @ w_mat).astype(np.float32)
    return _epilogue(y)
```

```cpp
#include <hip/hip_runtime.h>
#include <rccl/rccl.h>
#include <cstdio>

#define HIP_CHECK(expr) do { hipError_t _e = (expr); if (_e != hipSuccess) { \
    fprintf(stderr, "HIP error %s at %s:%d\n", hipGetErrorString(_e), __FILE__, __LINE__); \
    return; } } while (0)

#define LAS __attribute__((address_space(3)))
typedef short bf16x8 __attribute__((ext_vector_type(8)));
typedef short bf16x4 __attribute__((ext_vector_type(4)));
typedef float f32x4 __attribute__((ext_vector_type(4)));
typedef float f32x2 __attribute__((ext_vector_type(2)));
typedef __bf16 bf16x2v __attribute__((ext_vector_type(2)));
typedef unsigned u32x4 __attribute__((ext_vector_type(4)));
typedef unsigned short u16;

namespace {
constexpr int NR = 8;
constexpr int N_ = 16384;
constexpr int MB = 512, KB = 512;
constexpr int BM = 256, BN = 128, BK = 64;
constexpr int A_BYTES = BM * BK * 2, B_BYTES = BK * BN * 2, STAGE = A_BYTES + B_BYTES, NSTAGE = 3;
constexpr int LDS_CTRL = STAGE * NSTAGE;
constexpr int LDS_BYTES = LDS_CTRL + 64;
constexpr int NT = NR * KB / BK;
constexpr int NWG = 256;
constexpr int BLK_BYTES = MB * KB * 2;
constexpr size_t WS_CNT = 0;
constexpr size_t WS_TMO = 4096;
constexpr size_t WS_FLAG = 8192;
constexpr size_t WS_DATA = 1u << 20;
constexpr size_t WS_NEED = WS_DATA + 2ull * NR * BLK_BYTES;
constexpr unsigned long long DEADLINE = 100000000ull;
}

struct Args {
    const u16* x; const u16* w; u16* out;
    char* ws[NR];
    int rank;
};

__device__ __forceinline__ void glds16(unsigned ldsaddr, unsigned voff, const void* sbase) {
    asm volatile("s_mov_b32 m0, %0\n\ts_nop 0\n\tglobal_load_lds_dwordx4 %1, %2" :: "s"(ldsaddr), "v"(voff), "s"(sbase) : "memory");
}
#define BAR() do { asm volatile("" ::: "memory"); __builtin_amdgcn_s_barrier(); asm volatile("" ::: "memory"); } while (0)
#define WAIT_V(n) asm volatile("s_waitcnt vmcnt(" #n ")" ::: "memory")
#define WAIT_L(n) asm volatile("s_waitcnt lgkmcnt(" #n ")" ::: "memory")

__device__ __forceinline__ unsigned cvt_pk(float lo, float hi) {
    f32x2 f = {lo, hi}; bf16x2v b = __builtin_convertvector(f, bf16x2v); return __builtin_bit_cast(unsigned, b);
}
__device__ __forceinline__ unsigned vmcnt_now() {
    const unsigned st = __builtin_amdgcn_s_getreg(7 | (0 << 6) | (31 << 11));
    return (st & 15u) | (((st >> 22) & 3u) << 4);
}

__global__ __launch_bounds__(512, 2) void a2a_gemm(Args p) {
    extern __shared__ __attribute__((aligned(16))) char smem[];
    const int tid = threadIdx.x, lane = tid & 63;
    const int wid = __builtin_amdgcn_readfirstlane(tid >> 6);
    const int b = blockIdx.x, xcd = b & 7, idx = b >> 3;
    const int mh = idx & 1, nt = xcd * 16 + (idx >> 1);
    const int rank = p.rank;
    char* const myws = p.ws[rank];
    unsigned* const cntp = (unsigned*)(myws + WS_CNT) + b;
    const unsigned lds0 = (unsigned)(size_t)(LAS char*)smem;

    if (wid >= 4) {
        const int w = wid - 4;
        unsigned cv;
        asm volatile("global_load_dword %0, %1, off sc1" : "=v"(cv) : "v"(cntp) : "memory");
        unsigned voA[8], voB[4];
        { const int r = lane >> 3, c = lane & 7;
#pragma unroll
          for (int i = 0; i < 8; ++i) voA[i] = (unsigned)((mh * 256 + 64 * w + 8 * i + r) * (KB * 2) + ((c ^ r) * 16)); }
        { const int r = lane >> 4, c = lane & 15;
#pragma unroll
          for (int j = 0; j < 4; ++j) { const int row = 16 * w + 4 * j + r; const int f = (row & 3) | (((row >> 3) & 1) << 2);
              voB[j] = (unsigned)(row * (N_ * 2) + nt * 256 + ((((c >> 1) ^ f) << 1) | (c & 1)) * 16); } }
        const char* const xloc = (const char*)p.x + (size_t)rank * BLK_BYTES;
        const char* rbase = myws + WS_DATA;
        auto issue = [&](int t, int slot) {
            int src, kt; const char* ab;
            if (t < 8) { src = rank; kt = t; ab = xloc; }
            else { const int u = t - 8, q = u / 14, r = u - q * 14; src = (rank + 1 + (r >> 1)) & 7; kt = 2 * q + (r & 1); ab = rbase + (size_t)src * BLK_BYTES; }
            ab += kt * 128;
            const char* wb = (const char*)p.w + (size_t)(src * KB + kt * BK) * (N_ * 2);
            const unsigned la = lds0 + slot * STAGE + w * 8192, lb = lds0 + slot * STAGE + A_BYTES + w * 4096;
#pragma unroll
            for (int j = 0; j < 4; ++j) glds16(lb + j * 1024, voB[j], wb);
#pragma unroll
            for (int i = 0; i < 8; ++i) glds16(la + i * 1024, voA[i], ab);
        };
        if (w == 0 && lane == 0) *(LAS unsigned*)((LAS char*)smem + LDS_CTRL) = 0u;
        issue(0, 0); issue(1, 1); issue(2, 2);
        asm volatile("s_waitcnt vmcnt(24) lgkmcnt(0)" : "+v"(cv) :: "memory");
        const unsigned call = (unsigned)__builtin_amdgcn_readfirstlane(cv) + 1u;
        rbase += (size_t)(call & 1u) * (NR * BLK_BYTES);
        BAR();
        if (w == 0 && lane == 0) __hip_atomic_store(cntp, call, __ATOMIC_RELAXED, __HIP_MEMORY_SCOPE_AGENT);
        const unsigned* const fbase = (const unsigned*)(myws + WS_FLAG) + mh * 512 + lane * 8;
        const bool self = (lane >> 3) == rank;
        unsigned* const tmo = (unsigned*)(myws + WS_TMO);
        u32x4 f0 = {0, 0, 0, 0}, f1 = {0, 0, 0, 0};
        int slot = 0, nq = 0, tq = 8;
        for (int s = 1; s < NT - 2; ++s) {
            WAIT_V(12); BAR();
            if (s + 2 == tq) {
                asm volatile("" : "+v"(f0), "+v"(f1));
                const unsigned* fp = fbase + nq * 1024;
                bool ok = self || (f0[0] >= call && f0[1] >= call && f0[2] >= call && f0[3] >= call &&
                                   f1[0] >= call && f1[1] >= call && f1[2] >= call && f1[3] >= call);
                if (!__all(ok)) {
                    const unsigned long long t0 = __builtin_amdgcn_s_memrealtime();
                    for (;;) {
                        unsigned tv;
                        asm volatile("global_load_dwordx4 %0, %3, off sc0 sc1\n\tglobal_load_dwordx4 %1, %3, off offset:16 sc0 sc1\n\t"
                                     "global_load_dword %2, %4, off sc1\n\ts_waitcnt vmcnt(0)"
                                     : "=&v"(f0), "=&v"(f1), "=&v"(tv) : "v"(fp), "v"(tmo) : "memory");
                        ok = self || (f0[0] >= call && f0[1] >= call && f0[2] >= call && f0[3] >= call &&
                                      f1[0] >= call && f1[1] >= call && f1[2] >= call && f1[3] >= call);
                        if (__all(ok)) break;
                        if ((unsigned)__builtin_amdgcn_readfirstlane(tv) == call || __builtin_amdgcn_s_memrealtime() - t0 > DEADLINE) {
                            if (lane == 0) {
                                __hip_atomic_store(tmo, call, __ATOMIC_RELAXED, __HIP_MEMORY_SCOPE_AGENT);
                                *(LAS unsigned*)((LAS char*)smem + LDS_CTRL) = 1u;
                            }
                            WAIT_L(0);
                            break;
                        }
                        __builtin_amdgcn_s_sleep(2);
                    }
                }
                asm volatile("buffer_inv sc0 sc1" ::: "memory");
                ++nq; tq += 14;
            }
            issue(s + 2, slot); slot = slot == 2 ? 0 : slot + 1;
            if (s + 4 == tq && nq < 4) {
                const unsigned* fp = fbase + nq * 1024;
                asm volatile("global_load_dwordx4 %0, %2, off sc0 sc1\n\tglobal_load_dwordx4 %1, %2, off offset:16 sc0 sc1"
                             : "=&v"(f0), "=&v"(f1) : "v"(fp) : "memory");
            }
        }
        WAIT_V(12); BAR();
        WAIT_V(0); BAR();
        return;
    }

    const int wm = wid >> 1, wn = wid & 1;
    const int g = lane >> 4, l15 = lane & 15;
    int sent = 4;
    unsigned* flagp = nullptr;
    unsigned call = 0;
    if (b < 224) {
        const int dst = (rank + 1 + (b >> 5)) & 7, rg = b & 31;
        const int row = 16 * rg + 4 * wid + g;
        const char* sp = (const char*)p.x + (size_t)(dst * MB + row) * (KB * 2) + l15 * 16;
        u32x4 d0, d1, d2, d3; unsigned cv;
        asm volatile("global_load_dword %4, %6, off sc1\n\t"
                     "global_load_dwordx4 %0, %5, off\n\tglobal_load_dwordx4 %1, %5, off offset:256\n\t"
                     "global_load_dwordx4 %2, %5, off offset:512\n\tglobal_load_dwordx4 %3, %5, off offset:768\n\t"
                     "s_waitcnt vmcnt(0)"
                     : "=&v"(d0), "=&v"(d1), "=&v"(d2), "=&v"(d3), "=&v"(cv) : "v"(sp), "v"(cntp) : "memory");
        call = (unsigned)__builtin_amdgcn_readfirstlane(cv) + 1u;
        char* dws = p.ws[dst];
        char* dp = dws + WS_DATA + (size_t)((call & 1u) * NR + rank) * BLK_BYTES + (size_t)row * (KB * 2) + l15 * 16;
        asm volatile("global_store_dwordx4 %4, %0, off sc0 sc1\n\tglobal_store_dwordx4 %4, %1, off offset:256 sc0 sc1\n\t"
                     "global_store_dwordx4 %4, %2, off offset:512 sc0 sc1\n\tglobal_store_dwordx4 %4, %3, off offset:768 sc0 sc1\n\t"
                     "s_nop 1"
                     :: "v"(d0), "v"(d1), "v"(d2), "v"(d3), "v"(dp) : "memory");
        flagp = (unsigned*)(dws + WS_FLAG) + (rg >> 4) * 512 + rank * 64 + (rg & 15) * 4 + wid;
        sent = 0;
    }
    unsigned aoff[2], boff[4];
#pragma unroll
    for (int kk = 0; kk < 2; ++kk) aoff[kk] = (unsigned)((128 * wm + l15) * 128 + (((4 * kk + g) ^ (lane & 7)) * 16));
    { const int q = l15 >> 2, pp = lane & 3, f = q | ((g & 1) << 2);
#pragma unroll
      for (int j = 0; j < 4; ++j) boff[j] = (unsigned)(A_BYTES + (8 * g + q) * 256 + (((4 * wn + j) ^ f) * 32) + 8 * pp); }
    f32x4 acc[8][4];
#pragma unroll
    for (int i = 0; i < 8; ++i)
#pragma unroll
        for (int j = 0; j < 4; ++j) acc[i][j] = (f32x4){0.f, 0.f, 0.f, 0.f};
    bf16x8 fa0[8], fb0[4], fa1[8], fb1[4];
#define RD(FA, FB, so, kk) do { \
    _Pragma("unroll") for (int j = 0; j < 4; ++j) { \
      bf16x4 lo = __builtin_amdgcn_ds_read_tr16_b64_v4i16((LAS bf16x4*)((LAS char*)smem + (so) + boff[j] + (kk) * 8192)); \
      bf16x4 hi = __builtin_amdgcn_ds_read_tr16_b64_v4i16((LAS bf16x4*)((LAS char*)smem + (so) + boff[j] + (kk) * 8192 + 1024)); \
      FB[j] = __builtin_shufflevector(lo, hi, 0, 1, 2, 3, 4, 5, 6, 7); } \
    _Pragma("unroll") for (int i = 0; i < 8; ++i) FA[i] = *(const LAS bf16x8*)((LAS char*)smem + (so) + aoff[kk] + i * 2048); } while (0)
#define MM(FA, FB) do { \
    _Pragma("unroll") for (int i = 0; i < 8; ++i) _Pragma("unroll") for (int j = 0; j < 4; ++j) \
      acc[i][j] = __builtin_amdgcn_mfma_f32_16x16x32_bf16(FB[j], FA[i], acc[i][j], 0, 0, 0); } while (0)
#define SG() do { _Pragma("unroll") for (int z = 0; z < 8; ++z) { \
      __builtin_amdgcn_sched_group_barrier(0x8, 2, 0); __builtin_amdgcn_sched_group_barrier(0x100, 1, 0); \
      __builtin_amdgcn_sched_group_barrier(0x8, 1, 0); __builtin_amdgcn_sched_group_barrier(0x100, 1, 0); } \
      __builtin_amdgcn_sched_group_barrier(0x8, 8, 0); } while (0)
    __builtin_amdgcn_s_setprio(1);
    BAR();
    unsigned so = 0;
    RD(fa0, fb0, so, 0);
    for (int s = 0; s < NT - 1; ++s) {
        RD(fa1, fb1, so, 1);
        MM(fa0, fb0);
        SG();
        __builtin_amdgcn_sched_barrier(0);
        if (sent < 4) {
            if (s == 6 + 14 * sent) WAIT_V(3);
            if (vmcnt_now() <= 3u) {
                if (lane == 0) __hip_atomic_store(flagp + sent * 1024, call, __ATOMIC_RELAXED, __HIP_MEMORY_SCOPE_SYSTEM);
                ++sent;
            }
        }
        __builtin_amdgcn_sched_barrier(0);
        WAIT_L(0);
        BAR();
        __builtin_amdgcn_sched_barrier(0);
        so = so == 2 * STAGE ? 0 : so + STAGE;
        RD(fa0, fb0, so, 0);
        MM(fa1, fb1);
        SG();
    }
    RD(fa1, fb1, so, 1);
    MM(fa0, fb0);
    SG();
    MM(fa1, fb1);
    const bool bad = *(const volatile LAS unsigned*)((LAS char*)smem + LDS_CTRL) != 0u;
    u16* orow = p.out + (size_t)(mh * 256 + 128 * wm + l15) * N_ + nt * 128 + 64 * wn + ((g & 1) * 16 + (g >> 1) * 8);
#pragma unroll
    for (int i = 0; i < 8; ++i) {
#pragma unroll
        for (int jp = 0; jp < 2; ++jp) {
            unsigned a0 = cvt_pk(acc[i][2 * jp][0], acc[i][2 * jp][1]), a1 = cvt_pk(acc[i][2 * jp][2], acc[i][2 * jp][3]);
            unsigned b0 = cvt_pk(acc[i][2 * jp + 1][0], acc[i][2 * jp + 1][1]), b1 = cvt_pk(acc[i][2 * jp + 1][2], acc[i][2 * jp + 1][3]);
            auto s0 = __builtin_amdgcn_permlane16_swap(a0, b0, false, false);
            auto s1 = __builtin_amdgcn_permlane16_swap(a1, b1, false, false);
            u32x4 v = {s0[0], s1[0], s0[1], s1[1]};
            if (bad) v = (u32x4){0x7fc07fc0u, 0x7fc07fc0u, 0x7fc07fc0u, 0x7fc07fc0u};
            *(u32x4*)(orow + (size_t)i * 16 * N_ + jp * 32) = v;
        }
    }
}

extern "C" void kernel_launch(int rank, int nranks, ncclComm_t comm,
                              hipStream_t stream,
                              void* const* d_in, const int* in_sizes, int n_in,
                              void* d_out, int out_size,
                              const int* in_dtypes, int out_dtype,
                              void* const* d_out_all,
                              void* const* d_ws_all, size_t ws_bytes) {
    if (nranks != NR || (unsigned)rank >= (unsigned)NR || n_in < 2 || ws_bytes < WS_NEED ||
        in_sizes[0] != NR * MB * KB || in_sizes[1] != NR * KB * N_ || out_size != MB * N_) {
        fprintf(stderr, "a2a_gemm: unexpected problem shape\n");
        return;
    }
    static bool init = false;
    if (!init) {
        HIP_CHECK(hipFuncSetAttribute((const void*)a2a_gemm, hipFuncAttributeMaxDynamicSharedMemorySize, LDS_BYTES));
        init = true;
    }
    Args a;
    a.x = (const u16*)d_in[0]; a.w = (const u16*)d_in[1]; a.out = (u16*)d_out; a.rank = rank;
    for (int i = 0; i < NR; ++i) a.ws[i] = (char*)d_ws_all[i];
    hipLaunchKernelGGL(a2a_gemm, dim3(NWG), dim3(512), LDS_BYTES, stream, a);
    HIP_CHECK(hipGetLastError());
}
```

```cpp
#include <hip/hip_runtime.h>
#include <rccl/rccl.h>
#include <cstdio>

#define HIP_CHECK(expr) do { hipError_t _e = (expr); if (_e != hipSuccess) { \
    fprintf(stderr, "HIP error %s at %s:%d\n", hipGetErrorString(_e), __FILE__, __LINE__); \
    return; } } while (0)

#define LAS __attribute__((address_space(3)))
typedef short bf16x8 __attribute__((ext_vector_type(8)));
typedef short bf16x4 __attribute__((ext_vector_type(4)));
typedef float f32x4 __attribute__((ext_vector_type(4)));
typedef float f32x2 __attribute__((ext_vector_type(2)));
typedef __bf16 bf16x2v __attribute__((ext_vector_type(2)));
typedef unsigned u32x4 __attribute__((ext_vector_type(4)));
typedef unsigned u32x2 __attribute__((ext_vector_type(2)));
typedef unsigned short u16;

namespace {
constexpr int NR = 8;
constexpr int N_ = 16384;
constexpr int MB = 512, KB = 512;
constexpr int BM = 256, BN = 128, BK = 64;
constexpr int A_BYTES = BM * BK * 2;
constexpr int WH_BYTES = 32 * BN * 2;
constexpr int LDS_WL = 3 * A_BYTES, LDS_WE = LDS_WL + 3 * WH_BYTES;
constexpr int LDS_CTRL = LDS_WE + 4 * WH_BYTES;
constexpr int LDS_CNT = LDS_CTRL + 512;
constexpr int LDS_JUNK = LDS_CTRL + 1024;
constexpr int LDS_POLL = LDS_CTRL + 2048;
constexpr int LDS_BYTES = LDS_CTRL + 4096;
constexpr int NT = NR * KB / BK;
constexpr int NWG = 256;
constexpr int BLK_BYTES = MB * KB * 2;
constexpr size_t WS_CNT = 0;
constexpr size_t WS_TMO = 4096;
constexpr size_t WS_FLAG = 8192;
constexpr size_t WS_DATA = 1u << 20;
constexpr size_t WS_NEED = WS_DATA + 2ull * NR * BLK_BYTES;
constexpr unsigned long long DEADLINE = 100000000ull;
}

struct Args {
    const u16* x; const u16* w; u16* out; char* myws;
    char* ws[NR];
    int rank;
};

__device__ __forceinline__ const void* uptr(const void* q) {
    const unsigned long long v = (unsigned long long)q;
    const unsigned lo = __builtin_amdgcn_readfirstlane((unsigned)v), hi = __builtin_amdgcn_readfirstlane((unsigned)(v >> 32));
    return (const void*)(((unsigned long long)hi << 32) | lo);
}
__device__ __forceinline__ unsigned uval(unsigned v) { return __builtin_amdgcn_readfirstlane(v); }
__device__ __forceinline__ void glds16_plain(unsigned ldsaddr, unsigned voff, const void* sbase) {
    asm volatile("s_mov_b32 m0, %0\n\ts_nop 3\n\tglobal_load_lds_dwordx4 %1, %2" :: "s"(uval(ldsaddr)), "v"(voff), "s"(uptr(sbase)) : "memory");
}
__device__ __forceinline__ void glds16x4_sc1(unsigned ldsaddr, unsigned voff, const void* sbase) {
    asm volatile("s_mov_b32 m0, %0\n\ts_nop 0\n\tglobal_load_lds_dwordx4 %1, %2 sc1\n\tglobal_load_lds_dwordx4 %1, %2 offset:1024 sc1\n\t"
                 "global_load_lds_dwordx4 %1, %2 offset:2048 sc1\n\tglobal_load_lds_dwordx4 %1, %2 offset:3072 sc1"
                 :: "s"(uval(ldsaddr)), "v"(voff), "s"(uptr(sbase)) : "memory");
}
template <int PAD>
__device__ __forceinline__ void glds16w4(unsigned ldsaddr, unsigned v0, unsigned v1, unsigned v2, unsigned v3, const void* sbase) {
    asm volatile("s_mov_b32 m0, %0\n\ts_nop %6\n\tglobal_load_lds_dwordx4 %1, %5\n\tglobal_load_lds_dwordx4 %2, %5 offset:1024\n\t"
                 "global_load_lds_dwordx4 %3, %5 offset:2048\n\tglobal_load_lds_dwordx4 %4, %5 offset:3072"
                 :: "s"(uval(ldsaddr)), "v"(v0), "v"(v1), "v"(v2), "v"(v3), "s"(uptr(sbase)), "i"(PAD) : "memory");
}
__device__ __forceinline__ void glds4_sys(unsigned ldsaddr, unsigned voff, const void* sbase) {
    asm volatile("s_mov_b32 m0, %0\n\ts_nop 3\n\tglobal_load_lds_dword %1, %2 sc0 sc1\n\tglobal_load_lds_dword %1, %2 offset:256 sc0 sc1"
                 :: "s"(uval(ldsaddr)), "v"(voff), "s"(uptr(sbase)) : "memory");
}
__device__ __forceinline__ void glds4_sc1(unsigned ldsaddr, unsigned voff, const void* sbase) {
    asm volatile("s_mov_b32 m0, %0\n\ts_nop 3\n\tglobal_load_lds_dword %1, %2 sc1" :: "s"(uval(ldsaddr)), "v"(voff), "s"(uptr(sbase)) : "memory");
}
template <int V> struct IC { static constexpr int value = V; };
#define BAR() do { asm volatile("" ::: "memory"); __builtin_amdgcn_s_barrier(); asm volatile("" ::: "memory"); } while (0)
#define WAIT_V(n) asm volatile("s_waitcnt vmcnt(" #n ")" ::: "memory")
#define WAIT_L(n) asm volatile("s_waitcnt lgkmcnt(" #n ")" ::: "memory")

__device__ __forceinline__ unsigned cvt_pk(float lo, float hi) {
    f32x2 f = {lo, hi}; bf16x2v b = __builtin_convertvector(f, bf16x2v); return __builtin_bit_cast(unsigned, b);
}
__device__ __forceinline__ unsigned vmcnt_now() {
    const unsigned st = __builtin_amdgcn_s_getreg(7 | (0 << 6) | (31 << 11));
    return (st & 15u) | (((st >> 22) & 3u) << 4);
}


constexpr int PACE = 2;
constexpr int LPRIO = 3, CPRIO = 2;
constexpr unsigned long long SPIN = 700;

__global__ __launch_bounds__(512, 2) void a2a_gemm(Args p) {
    extern __shared__ __attribute__((aligned(16))) char smem[];
    const int tid = threadIdx.x, lane = tid & 63;
    const int wid = __builtin_amdgcn_readfirstlane(tid >> 6);
    const int b = blockIdx.x, xcd = b & 7, idx = b >> 3;
    const int mh = idx & 1, nt = xcd * 16 + (idx >> 1);
    const int colb0 = nt * 256, colbs = 128;
    const int rank = p.rank;
    char* const myws = p.myws;
    unsigned* const cntp = (unsigned*)(myws + WS_CNT) + b;
    const unsigned lds0 = (unsigned)(size_t)(LAS char*)smem;
    const char* const xloc = (const char*)p.x + (size_t)rank * BLK_BYTES;

    if (wid == 7) {
        __builtin_amdgcn_s_setprio(CPRIO);
        glds16_plain(lds0 + LDS_JUNK, (unsigned)(b * 2048 + lane * 16), xloc);
        glds16_plain(lds0 + LDS_JUNK, (unsigned)(b * 2048 + 1024 + lane * 16), xloc);
        if (b < 224) {
            const int dst = (rank + 1 + (b >> 5)) & 7, rg = b & 31;
            const u32x4* sp = (const u32x4*)((const char*)p.x + (size_t)(dst * MB + 16 * rg + (lane >> 3)) * (KB * 2) + (((lane & 7) ^ (lane >> 3)) * 16));
            u32x4 d[8][2]; unsigned cv;
            asm volatile("global_load_dword %2, %5, off sc1\n\tglobal_load_dwordx4 %0, %3, off\n\tglobal_load_dwordx4 %1, %4, off\n\ts_waitcnt vmcnt(0)"
                         : "=&v"(d[0][0]), "=&v"(d[0][1]), "=&v"(cv) : "v"(sp), "v"(sp + 512), "v"(cntp) : "memory");
            const unsigned call = (unsigned)__builtin_amdgcn_readfirstlane(cv) + 1u;
            char* dws = p.ws[dst];
            char* dp = dws + WS_DATA + (size_t)((call & 1u) * NR + rank) * BLK_BYTES + rg * 2048 + lane * 16;
            unsigned* flagp = (unsigned*)(dws + WS_FLAG) + (rg >> 4) * 128 + rank * 16 + (rg & 15);
            const int toff = ((b >> 3) & 3) - 1;
            int nb = 0;
#pragma unroll
            for (int r = 0; r < 8; ++r) {
                constexpr int FT[8] = {6, 13, 20, 27, 34, 41, 48, 55};
                while (nb <= PACE * r + toff && nb < FT[r]) { BAR(); ++nb; }
                asm volatile("global_store_dwordx4 %2, %0, off sc0 sc1\n\tglobal_store_dwordx4 %2, %1, off offset:1024 sc0 sc1\n\ts_nop 1"
                             :: "v"(d[r][0]), "v"(d[r][1]), "v"(dp + r * 65536) : "memory");
                if (r == 0) {
#pragma unroll
                    for (int q = 1; q < 8; ++q) { d[q][0] = sp[q * 8]; d[q][1] = sp[q * 8 + 512]; }
                    asm volatile("" ::: "memory");
                }
                while (vmcnt_now() != 0u) {
                    if (nb >= FT[r]) { WAIT_V(0); break; }
                    const unsigned long long c0 = __builtin_amdgcn_s_memtime();
                    bool done = false;
                    while (__builtin_amdgcn_s_memtime() - c0 < SPIN) { if (vmcnt_now() == 0u) { done = true; break; } __builtin_amdgcn_s_sleep(1); }
                    if (done) break;
                    BAR(); ++nb;
                }
                if (lane == 0) __hip_atomic_store(flagp + r * 256, call, __ATOMIC_RELAXED, __HIP_MEMORY_SCOPE_SYSTEM);
            }
        }
        return;
    }
    if (wid >= 4) {
        __builtin_amdgcn_s_setprio(LPRIO);
        auto loader = [&](auto WC) {
        constexpr int w = decltype(WC)::value;
        constexpr int NA = w == 2 ? 16 : 8, afirst = 8 * w;
        unsigned vw[4], va[NA];
        { const int r = lane >> 4, c = lane & 15;
#pragma unroll
          for (int j = 0; j < 4; ++j) { const int row = 16 * (w & 1) + 4 * j + r; const int f = (row & 3) | (((row >> 3) & 1) << 2);
              const int u = (c >> 1) ^ f;
              vw[j] = (unsigned)(row * (N_ * 2) + colb0 + (u >> 2) * colbs + (u & 3) * 32 + (c & 1) * 16 + 4096 - j * 1024); } }
        { const int r = lane >> 3, c = lane & 7;
#pragma unroll
          for (int i = 0; i < NA; ++i) va[i] = (unsigned)((mh * 256 + 8 * (afirst + i) + r) * (KB * 2) + ((c ^ r) * 16)); }
        const unsigned voR = (unsigned)((mh * 32 + afirst) * 1024 + lane * 16);
        const char* rbase = myws + WS_DATA;
        const char* const we = (const char*)p.w + (size_t)(mh * 32) * (N_ * 2) - 4096;
        const char* const wl = (const char*)p.w + (size_t)((mh ^ 1) * 32) * (N_ * 2) - 4096;
        const unsigned lwl = lds0 + LDS_WL + (w & 1) * 4096, lwe = lds0 + LDS_WE + (w & 1) * 4096, lal = lds0 + afirst * 1024;
        auto issue_w = [&](const char* wb, unsigned lb) { if constexpr (w != 2) glds16w4<3>(lb, vw[0], vw[1], vw[2], vw[3], wb); };
        auto issue_wh = [&](const char* wb, unsigned lb) { if constexpr (w != 2) glds16w4<0>(lb, vw[0], vw[1], vw[2], vw[3], wb); };
        auto issue_al = [&](int t, int slot) {
#pragma unroll
            for (int i = 0; i < NA; ++i) glds16_plain(lal + slot * A_BYTES + i * 1024, va[i], xloc + t * 128); };
        auto issue_ar = [&](const char* ab, unsigned la) {
#pragma unroll
            for (int i = 0; i < NA / 4; ++i) glds16x4_sc1(la + i * 4096, voR, ab + i * 4096); };
        volatile LAS unsigned* const sched = (volatile LAS unsigned*)((LAS char*)smem + LDS_CTRL + 64);
        volatile LAS unsigned* const ctl = (volatile LAS unsigned*)((LAS char*)smem + LDS_CTRL + 4);
        int vs = lane < 8 ? rank * 8 + lane : ((rank + 1 + (lane - 8) % 7) & 7) * 8 + (lane - 8) / 7;
        int chk = 6;
        auto sk_of = [&](int t) -> unsigned { return (unsigned)__builtin_amdgcn_readlane(vs, t) & 63u; };
        constexpr size_t WT = (size_t)BK * N_ * 2;
#define WAIT_STAGE() do { if constexpr (w == 2) WAIT_V(16); else WAIT_V(20); } while (0)
#pragma unroll
        for (int t = 0; t < 3; ++t) {
            issue_w(we + (size_t)(rank * 8 + t) * WT, lwe + t * WH_BYTES); issue_w(wl + (size_t)(rank * 8 + t) * WT, lwl + t * WH_BYTES); issue_al(t, t);
        }
        issue_w(we + (size_t)(rank * 8 + 3) * WT, lwe + 3 * WH_BYTES);
        if constexpr (w == 0) glds4_sc1(lds0 + LDS_CNT, 0u, cntp);
        if (w == 0 && lane < 4) ((LAS unsigned*)((LAS char*)smem + LDS_CTRL))[lane] = 0u;
        if constexpr (w == 2) WAIT_V(32); else asm volatile("s_waitcnt vmcnt(36) lgkmcnt(0)" ::: "memory");
        BAR();
#pragma unroll
        for (int s = 1; s < 5; ++s) {
            WAIT_STAGE(); BAR();
            issue_w(wl + (size_t)(rank * 8 + s + 2) * WT, lwl + ((s + 2) % 3) * WH_BYTES); issue_al(s + 2, (s + 2) % 3);
            issue_w(we + (size_t)(rank * 8 + s + 3) * WT, lwe + ((s + 3) & 3) * WH_BYTES);
        }
        const unsigned call = uval(*(volatile LAS unsigned*)((LAS char*)smem + LDS_CNT)) + 1u;
        rbase += (size_t)(call & 1u) * (NR * BLK_BYTES);
        const char* const fp = myws + WS_FLAG + mh * 512;
        unsigned* const tmo = (unsigned*)(myws + WS_TMO);
        const int ck = lane >> 3, csk = ((rank + 1 + (lane & 7)) & 7) * 8 + ck;
        constexpr unsigned long long ALL = 0x7f7f7f7f7f7f7f7full;
        unsigned long long okm = 0, todo = ALL;
        int pdue = -1; unsigned pr0 = 0, pr1 = 0, pr2 = 0;
        int hi = 0;
        auto j_of = [&](int t) -> unsigned { const unsigned sk = sk_of(t); return (sk & 7u) * 8u + (((sk >> 3) - rank - 1) & 7u); };
        auto digest1 = [&](int slot_, unsigned k) {
            const u32x2 f = *(const volatile LAS u32x2*)((LAS char*)smem + LDS_POLL + slot_ * 512 + lane * 8);
            const unsigned long long miss = __ballot(f[0] < call || f[1] < call);
            unsigned r = 0;
#pragma unroll
            for (int i = 0; i < 7; ++i) r |= ((miss >> (8 * ((rank + 1 + i) & 7))) & 0xffull) ? 0u : (1u << i);
            okm |= (unsigned long long)r << (8 * k);
        };
        auto digest = [&]() { digest1(0, pr0); digest1(1, pr1); digest1(2, pr2); };
        auto rows = [&](unsigned lo, bool wide) {
            pr0 = lo; pr1 = wide ? min(lo + 1u, 7u) : lo; pr2 = wide ? min((unsigned)hi + 1u, 7u) : lo; };
        auto poll3 = [&]() { glds4_sys(lds0 + LDS_POLL, lane * 4u, fp + pr0 * 1024); glds4_sys(lds0 + LDS_POLL + 512, lane * 4u, fp + pr1 * 1024);
                             glds4_sys(lds0 + LDS_POLL + 1024, lane * 4u, fp + pr2 * 1024); };
        auto rankof = [&](unsigned long long m) { return (int)__builtin_amdgcn_mbcnt_hi((unsigned)(m >> 32), __builtin_amdgcn_mbcnt_lo((unsigned)m, 0u)); };
        auto validate = [&](int s) -> bool {
            const int t = s + 3;
            bool redo = false;
            unsigned j = j_of(t);
            const bool cp = s + 1 == chk;
            if (!((okm >> j) & 1ull) || cp) {
                const unsigned long long t0 = __builtin_amdgcn_s_memrealtime();
                while (cp ? !(okm & todo) : !((okm >> j) & 1ull)) {
                    if (pdue >= 0) { WAIT_V(0); digest(); pdue = -1; }
                    rows(j >> 3, cp && s > 5);
                    __builtin_amdgcn_s_setprio(0);
                    const unsigned tv = __hip_atomic_load(tmo, __ATOMIC_RELAXED, __HIP_MEMORY_SCOPE_AGENT);
                    poll3(); WAIT_V(0);
                    digest();
                    if ((unsigned)__builtin_amdgcn_readfirstlane(tv) == call || __builtin_amdgcn_s_memrealtime() - t0 > DEADLINE) {
                        if (lane == 0) {
                            __hip_atomic_store(tmo, call, __ATOMIC_RELAXED, __HIP_MEMORY_SCOPE_AGENT);
                            *(LAS unsigned*)((LAS char*)smem + LDS_CTRL) = 1u;
                        }
                        okm = ALL;
                        break;
                    }
                }
                __builtin_amdgcn_s_setprio(LPRIO);
                if (cp) {
                    const unsigned long long A = okm & todo, B = todo & ~okm;
                    const int na = __builtin_popcountll(A);
                    hi = max(hi, (63 - __builtin_clzll(A)) >> 3);
                    const bool aligned = !B || (int)(__builtin_ctzll(B) >> 3) > hi;
                    const bool mine = (todo >> lane) & 1ull;
                    if (mine) sched[t + (((A >> lane) & 1ull) ? rankof(A) : na + rankof(B))] = (unsigned)csk;
                    WAIT_L(0);
                    const int old = vs;
                    if (lane >= t) vs = (int)sched[lane];
                    sched[lane] = (unsigned)vs;
                    const unsigned jn = j_of(t);
                    redo = jn != j; j = jn;
                    chk = aligned ? 0 : t + na - 2;
                    const unsigned c = (__any(old != vs) ? 1u : 0u) | (redo ? 2u : 0u) | (unsigned)chk << 8;
                    if (lane == 0) ctl[s & 1] = c;
                }
                WAIT_L(0);
            }
            todo &= ~(1ull << j);
            return redo;
        };
        bool redo = false;
        if constexpr (w == 0) {
            if (lane == 0) __hip_atomic_store(cntp, call, __ATOMIC_RELAXED, __HIP_MEMORY_SCOPE_AGENT);
            glds4_sys(lds0 + LDS_POLL, lane * 4u, fp);
        }
        {
            WAIT_STAGE(); BAR();
            issue_w(wl + (size_t)(rank * 8 + 7) * WT, lwl + 1 * WH_BYTES); issue_al(7, 1);
            issue_w(we + (size_t)sk_of(8) * WT, lwe);
            if constexpr (w == 0) {
                WAIT_V(16);
                digest1(0, 0);
                redo = validate(5);
            }
        }
        int slot = 2;
        for (int s = 6; s < NT - 2; ++s) {
            unsigned ska = sk_of(s + 2), ske = sk_of(s + 3 < NT ? s + 3 : s + 2);
            const char* pwl = wl + (size_t)ska * WT; const char* pa = rbase + (size_t)ska * 65536; const char* pwe = we + (size_t)ske * WT;
            unsigned dl = lwl + slot * WH_BYTES, da = lal + slot * A_BYTES, de = lwe + ((s + 3) & 3) * WH_BYTES;
            asm volatile("" : "+s"(pwl), "+s"(pa), "+s"(pwe), "+s"(dl), "+s"(da), "+s"(de));
            WAIT_STAGE();
            BAR();
            if constexpr (w != 0) {
                if (s == chk) {
                    const unsigned c = uval(ctl[(s - 1) & 1]);
                    if (c & 1u) {
                        vs = (int)sched[lane];
                        ska = sk_of(s + 2); ske = sk_of(s + 3 < NT ? s + 3 : s + 2);
                        pwl = wl + (size_t)ska * WT; pa = rbase + (size_t)ska * 65536; pwe = we + (size_t)ske * WT;
                    }
                    redo = c & 2u; chk = (int)(c >> 8);
                }
            }
            issue_wh(pwl, dl); issue_ar(pa, da);
            if (redo) { issue_w(we + (size_t)ska * WT, lwe + ((s + 2) & 3) * WH_BYTES); redo = false; }
            if constexpr (w == 0) {
                if (s == pdue) { digest(); pdue = -1; }
                if (pdue < 0 && s + 5 < NT) {
                    const unsigned jn = j_of(s + 5);
                    if (!((okm >> jn) & 1ull)) {
                        rows(jn >> 3, s + 3 == chk);
                        poll3();
                        pdue = s + 2;
                    }
                }
            }
            if (s + 3 < NT) issue_wh(pwe, de);
            slot = slot == 2 ? 0 : slot + 1;
            if constexpr (w == 0) { if (s + 3 < NT) redo = validate(s); }
        }
        WAIT_V(16); BAR();
        WAIT_V(0); BAR();
        };
        if (wid == 4) loader(IC<0>{}); else if (wid == 5) loader(IC<1>{}); else loader(IC<2>{});
        return;
    }

    const int wm = wid >> 1, wn = wid & 1;
    const int g = lane >> 4, l15 = lane & 15;
    unsigned aoff[2], boff[4];
#pragma unroll
    for (int kk = 0; kk < 2; ++kk) aoff[kk] = (unsigned)((128 * wm + l15) * 128 + (((4 * kk + g) ^ (lane & 7)) * 16));
    { const int q = l15 >> 2, pp = lane & 3, f = q | ((g & 1) << 2);
#pragma unroll
      for (int j = 0; j < 4; ++j) boff[j] = (unsigned)((8 * g + q) * 256 + (((4 * wn + j) ^ f) * 32) + 8 * pp); }
    f32x4 acc[8][4];
#pragma unroll
    for (int i = 0; i < 8; ++i)
#pragma unroll
        for (int j = 0; j < 4; ++j) acc[i][j] = (f32x4){0.f, 0.f, 0.f, 0.f};
    bf16x8 fa0[8], fb0[4], fa1[8], fb1[4];
#define RD(FA, FB, so, wo, kk) do { \
    _Pragma("unroll") for (int j = 0; j < 4; ++j) { \
      bf16x4 lo = __builtin_amdgcn_ds_read_tr16_b64_v4i16((LAS bf16x4*)((LAS char*)smem + (wo) + boff[j])); \
      bf16x4 hi = __builtin_amdgcn_ds_read_tr16_b64_v4i16((LAS bf16x4*)((LAS char*)smem + (wo) + boff[j] + 1024)); \
      FB[j] = __builtin_shufflevector(lo, hi, 0, 1, 2, 3, 4, 5, 6, 7); } \
    _Pragma("unroll") for (int i = 0; i < 8; ++i) FA[i] = *(const LAS bf16x8*)((LAS char*)smem + (so) + aoff[kk] + i * 2048); } while (0)
#define MM(FA, FB) do { \
    _Pragma("unroll") for (int i = 0; i < 8; ++i) _Pragma("unroll") for (int j = 0; j < 4; ++j) \
      acc[i][j] = __builtin_amdgcn_mfma_f32_16x16x32_bf16(FB[j], FA[i], acc[i][j], 0, 0, 0); } while (0)
#define SG() do { _Pragma("unroll") for (int z = 0; z < 8; ++z) { \
      __builtin_amdgcn_sched_group_barrier(0x8, 2, 0); __builtin_amdgcn_sched_group_barrier(0x100, 1, 0); \
      __builtin_amdgcn_sched_group_barrier(0x8, 1, 0); __builtin_amdgcn_sched_group_barrier(0x100, 1, 0); } \
      __builtin_amdgcn_sched_group_barrier(0x8, 8, 0); } while (0)
#define WOFF(kk, s3, s4) (unsigned)(((kk) == mh) ? LDS_WE + (s4) * WH_BYTES : LDS_WL + (s3) * WH_BYTES)
#define STEP(cur, nxt) do { \
        RD(fa1, fb1, (cur) * A_BYTES, WOFF(1, cur, e4), 1); MM(fa0, fb0); SG(); \
        __builtin_amdgcn_sched_barrier(0); WAIT_L(0); BAR(); __builtin_amdgcn_sched_barrier(0); \
        e4 = (e4 + 1) & 3; \
        RD(fa0, fb0, (nxt) * A_BYTES, WOFF(0, nxt, e4), 0); MM(fa1, fb1); SG(); __builtin_amdgcn_sched_barrier(0); } while (0)
    __builtin_amdgcn_s_setprio(1);
    BAR();
    int e4 = 0;
    RD(fa0, fb0, 0, WOFF(0, 0, 0), 0);
    for (int s = 0; s < (NT - 1) / 3; ++s) {
        STEP(0, 1); STEP(1, 2); STEP(2, 0);
    }
    RD(fa1, fb1, 0, WOFF(1, 0, e4), 1);
    MM(fa0, fb0);
    SG();
    MM(fa1, fb1);
    const bool bad = *(const volatile LAS unsigned*)((LAS char*)smem + LDS_CTRL) != 0u;
    u16* orow = p.out + (size_t)(mh * 256 + 128 * wm + l15) * N_ + (colb0 + wn * colbs) / 2 + ((g & 1) * 16 + (g >> 1) * 8);
#pragma unroll
    for (int i = 0; i < 8; ++i) {
#pragma unroll
        for (int jp = 0; jp < 2; ++jp) {
            unsigned a0 = cvt_pk(acc[i][2 * jp][0], acc[i][2 * jp][1]), a1 = cvt_pk(acc[i][2 * jp][2], acc[i][2 * jp][3]);
            unsigned b0 = cvt_pk(acc[i][2 * jp + 1][0], acc[i][2 * jp + 1][1]), b1 = cvt_pk(acc[i][2 * jp + 1][2], acc[i][2 * jp + 1][3]);
            auto s0 = __builtin_amdgcn_permlane16_swap(a0, b0, false, false);
            auto s1 = __builtin_amdgcn_permlane16_swap(a1, b1, false, false);
            u32x4 v = {s0[0], s1[0], s0[1], s1[1]};
            if (bad) v = (u32x4){0x7fc07fc0u, 0x7fc07fc0u, 0x7fc07fc0u, 0x7fc07fc0u};
            *(u32x4*)(orow + (size_t)i * 16 * N_ + jp * 32) = v;
        }
    }
}


extern "C" void kernel_launch(int rank, int nranks, ncclComm_t comm,
                              hipStream_t stream,
                              void* const* d_in, const int* in_sizes, int n_in,
                              void* d_out, int out_size,
                              const int* in_dtypes, int out_dtype,
                              void* const* d_out_all,
                              void* const* d_ws_all, size_t ws_bytes) {
    if (nranks != NR || (unsigned)rank >= (unsigned)NR || n_in < 2 || ws_bytes < WS_NEED ||
        in_sizes[0] != NR * MB * KB || in_sizes[1] != NR * KB * N_ || out_size != MB * N_) {
        fprintf(stderr, "a2a_gemm: unexpected problem shape\n");
        return;
    }
    static bool init = false;
    if (!init) {
        HIP_CHECK(hipFuncSetAttribute((const void*)a2a_gemm, hipFuncAttributeMaxDynamicSharedMemorySize, LDS_BYTES));
        init = true;
    }
    Args a;
    a.x = (const u16*)d_in[0]; a.w = (const u16*)d_in[1]; a.out = (u16*)d_out; a.rank = rank; a.myws = (char*)d_ws_all[rank];
    for (int i = 0; i < NR; ++i) a.ws[i] = (char*)d_ws_all[i];
    hipLaunchKernelGGL(a2a_gemm, dim3(NWG), dim3(512), LDS_BYTES, stream, a);
    HIP_CHECK(hipGetLastError());
}
```
